```python
import numpy as np

kernel_name = "gpu_ag_gemm_m4096_k4096_n4096_f32_reluq8"


ABSMAX_RTOL = 0.02

COMPUTE_DTYPE = "float32"
EPILOGUE = "relu_quant_int8"
IN_QUANT = None
OUT_QUANT = "int8"
IN_DTYPES = {"x": "float32", "w_mat": "float32"}
WORLD_SIZE = 8

def setup_inputs():

    x = np.random.randn(4096, 4096).astype(np.float32)
    w_mat = (np.random.randn(4096, 4096) / np.sqrt(4096)).astype(np.float32)
    return {"x": x, "w_mat": w_mat}

def _quant_dequant(y, amax):
    scale = amax / 127.0
    q = np.clip(np.round(y / scale), -127.0, 127.0)
    return (q * scale).astype(np.float32)
def _epilogue(y):
    y = np.maximum(y, 0.0)
    return _quant_dequant(y, np.abs(y).max())

def reference(x, w_mat):
    y = (x @ w_mat).astype(np.float32)
    return _epilogue(y)
```

```cpp
#include <hip/hip_runtime.h>
#include <rccl/rccl.h>
#include <cstdio>

#define HIP_CHECK(expr) do { hipError_t _e = (expr); if (_e != hipSuccess) { \
    fprintf(stderr, "HIP error %s at %s:%d\n", hipGetErrorString(_e), __FILE__, __LINE__); \
    return; } } while (0)

namespace {

typedef __attribute__((ext_vector_type(4))) unsigned u32x4;
typedef __attribute__((ext_vector_type(2))) unsigned u32x2;
typedef __attribute__((ext_vector_type(4))) float f32x4;
typedef __attribute__((ext_vector_type(8))) __bf16 bf16x8;

#define NWL 20
#define NWLOC 104
#define NSB 10
#define LINK_PER_WG 0
#define GPW 3
#define LDS_PAD (84 * 1024)
#define AUX_LD 17
#define POLL_SLEEP 4

constexpr int NR = 8;
constexpr int KDIM = 4096, NLOC = 512;
constexpr unsigned STREAM_BYTES = 512u * 1024u;
constexpr unsigned STREAM_STRIDE = STREAM_BYTES + 4096u;
constexpr unsigned NSTREAM = 48;
constexpr size_t FLAG_OFF = 32ull << 20;
constexpr unsigned FLAG_STRIDE = 4352;
constexpr size_t CTRL_OFF = 33ull << 20;
constexpr size_t CNT_OFF = CTRL_OFF;
constexpr size_t TMO_OFF = CTRL_OFF + 256;
constexpr size_t TICK_OFF = CTRL_OFF + 1024;
constexpr size_t G1_OFF = CTRL_OFF + 8192;
constexpr size_t G2_OFF = CTRL_OFF + 65536;
constexpr size_t DONE_OFF = CTRL_OFF + 196608;
constexpr size_t WS_NEED = CTRL_OFF + 262144;
constexpr int AUX_SYS = 17;
constexpr int RSRC_W3 = 0x00020000;
constexpr unsigned long long DEADLINE_TICKS = 100000000ull;
constexpr unsigned long long TMO_LOOK_TICKS = 20000ull;

#define PROF_T(slot) do { } while (0)

#define DBG_SO(x) (x)
struct Params {
  const float* x;
  const float* w;
  char* ws[NR];
  char* out[NR];
  int rank;
};

__device__ __forceinline__ unsigned pk_bf16(float a, float b) {
  unsigned r;
  asm("v_cvt_pk_bf16_f32 %0, %1, %2" : "=v"(r) : "v"(a), "v"(b));
  return r;
}
__device__ __forceinline__ unsigned ld_sys32(const void* p) {
  return __hip_atomic_load((const unsigned*)p, __ATOMIC_RELAXED, __HIP_MEMORY_SCOPE_SYSTEM);
}
__device__ __forceinline__ unsigned long long ld_sys64(const void* p) {
  return __hip_atomic_load((const unsigned long long*)p, __ATOMIC_RELAXED, __HIP_MEMORY_SCOPE_SYSTEM);
}
__device__ __forceinline__ void st_sys32(void* p, unsigned v) {
  __hip_atomic_store((unsigned*)p, v, __ATOMIC_RELAXED, __HIP_MEMORY_SCOPE_SYSTEM);
}
__device__ __forceinline__ void st_sys64(void* p, unsigned long long v) {
  __hip_atomic_store((unsigned long long*)p, v, __ATOMIC_RELAXED, __HIP_MEMORY_SCOPE_SYSTEM);
}

__device__ __forceinline__ bool wait_expired(unsigned long long t0, char* myws, unsigned seq) {
  const unsigned long long el = __builtin_amdgcn_s_memrealtime() - t0;
  if (el > TMO_LOOK_TICKS) {
    unsigned* tmo = (unsigned*)(myws + TMO_OFF);
    const unsigned tv = __builtin_amdgcn_readfirstlane(
        __hip_atomic_load(tmo, __ATOMIC_RELAXED, __HIP_MEMORY_SCOPE_AGENT));
    if (tv == seq || el > DEADLINE_TICKS) {
      __hip_atomic_store(tmo, seq, __ATOMIC_RELAXED, __HIP_MEMORY_SCOPE_AGENT);
      return true;
    }
  }
  return false;
}

__device__ __forceinline__ unsigned wave_max_u(unsigned v) {
  v = max(v, (unsigned)__builtin_amdgcn_update_dpp(0, (int)v, 0x111, 0xf, 0xf, false));
  v = max(v, (unsigned)__builtin_amdgcn_update_dpp(0, (int)v, 0x112, 0xf, 0xf, false));
  v = max(v, (unsigned)__builtin_amdgcn_update_dpp(0, (int)v, 0x114, 0xf, 0xf, false));
  v = max(v, (unsigned)__builtin_amdgcn_update_dpp(0, (int)v, 0x118, 0xf, 0xf, false));
  v = max(v, (unsigned)__builtin_amdgcn_update_dpp(0, (int)v, 0x142, 0xa, 0xf, false));
  v = max(v, (unsigned)__builtin_amdgcn_update_dpp(0, (int)v, 0x143, 0xc, 0xf, false));
  return (unsigned)__builtin_amdgcn_readlane((int)v, 63);
}

}

__global__ void __launch_bounds__(256, 2) ag_gemm_fused(Params P) {
  const int lane = threadIdx.x & 63;
  const int wv = __builtin_amdgcn_readfirstlane(threadIdx.x >> 6);
  const int blk = blockIdx.x;
  const unsigned rank = (unsigned)P.rank & 7u;
  char* const myws = P.ws[rank];
  const int l15 = lane & 15, q = lane >> 4;
  PROF_T(0);

  const int xg = blk & 7, li = blk >> 3;
  const int ti = (li - NSB) * GPW + wv;
  if (li < NSB || wv >= GPW || ti >= 64) {
    int sw;
    if (li < NSB) { sw = blk * 4 + wv; if (sw >= 7 * NWL) return; }
    else if (wv >= GPW) { sw = 7 * NWL + (blk - 8 * NSB); if (wv > GPW || sw >= 7 * NWL + NWLOC) return; }
    else { if (ti != 64) return; sw = -1; }
    const unsigned cnt = __hip_atomic_load((const unsigned*)(myws + CNT_OFF), __ATOMIC_RELAXED, __HIP_MEMORY_SCOPE_AGENT);
    if (sw >= 0) {
      const bool loc = sw >= 7 * NWL;
      const int d = loc ? 0 : 1 + sw % 7;
      int t = loc ? sw - 7 * NWL : sw / 7;
      const int np = loc ? 13 * 64 : 5 * 64;
      const int j = (d >> 1) - 1;
      const unsigned peer = (rank ^ (unsigned)d) & 7u;
      char* const pws = P.ws[peer];
      const __amdgpu_buffer_rsrc_t rd = __builtin_amdgcn_make_buffer_rsrc(pws, 0, (int)(NSTREAM * STREAM_STRIDE), RSRC_W3);
      const __amdgpu_buffer_rsrc_t rf = __builtin_amdgcn_make_buffer_rsrc(pws + FLAG_OFF, 0, (int)(NSTREAM * FLAG_STRIDE), RSRC_W3);
      const __amdgpu_buffer_rsrc_t rt = __builtin_amdgcn_make_buffer_rsrc(myws + TICK_OFF + d * 256, 0, loc ? 0 : 4, RSRC_W3);
      const int foff = lane * (1 << 20);
      const int voffx = (l15 * KDIM + 4 * q) * 4, voffw = ((4 * q) * NLOC + 4 * l15) * 4;
      bool isx;
      int ks, stream;
      u32x4 ld[16];
#define DECODE_LOAD(tt) { \
        int sub, src0; \
        if (loc) { ks = (tt) / 13; sub = (tt) - 13 * ks; isx = sub < 5; \
                   src0 = isx ? 64 * sub : 64 * (sub - 5); stream = isx ? 5 * (rank & 1) + sub : 16 + sub - 5; } \
        else { ks = (tt) / 5; sub = (tt) - 5 * ks; isx = (d == 1) || (sub == 0); \
               src0 = (d == 1) ? 64 * sub : (sub == 0 ? 320 + 64 * j : 256 * (d & 1) + 64 * (sub - 1)); \
               stream = (d == 1) ? 5 * (rank & 1) + sub : (sub == 0 ? 10 + 2 * j + (rank & 1) : 24 + 4 * (2 * j + (rank & 1)) + sub - 1); } \
        const __amdgpu_buffer_rsrc_t rs = __builtin_amdgcn_make_buffer_rsrc((void*)(isx ? P.x : P.w), 0, 512 * 4096 * 4, RSRC_W3); \
        const int vo_ = isx ? voffx : voffw; \
        const int so_ = DBG_SO(isx ? src0 * (KDIM * 4) + ks * 256 : src0 * 4 + ks * (64 * NLOC * 4)); \
        const int sa_ = isx ? 64 : NLOC * 4, sc_ = isx ? 16 * KDIM * 4 : 16 * NLOC * 4; \
        _Pragma("unroll") for (int i = 0; i < 16; ++i) \
          ld[i] = __builtin_amdgcn_raw_buffer_load_b128(rs, vo_, so_ + (i & 3) * sa_ + (i >> 2) * sc_, 0); \
      }
      DECODE_LOAD(t);
      asm volatile("" ::: "memory");
      __builtin_amdgcn_raw_buffer_store_b32(0u, rf, foff + (1 << 20), 0, AUX_SYS);
      const unsigned seq = __builtin_amdgcn_readfirstlane(cnt) + 1u;
      const unsigned tbase = (unsigned)NWL - (seq - 1u) * (unsigned)np;
      for (;;) {
        u32x4 o[8];
        if (isx) {
#pragma unroll
          for (int kh = 0; kh < 2; ++kh)
#pragma unroll
            for (int mb = 0; mb < 4; ++mb) {
              const f32x4 a = __builtin_bit_cast(f32x4, ld[mb * 4 + 2 * kh]);
              const f32x4 b = __builtin_bit_cast(f32x4, ld[mb * 4 + 2 * kh + 1]);
              o[kh * 4 + mb] = u32x4{pk_bf16(a.x, a.y), pk_bf16(a.z, a.w), pk_bf16(b.x, b.y), pk_bf16(b.z, b.w)};
            }
        } else {
#pragma unroll
          for (int kh = 0; kh < 2; ++kh)
#pragma unroll
            for (int b = 0; b < 4; ++b) {
              unsigned r[4];
#pragma unroll
              for (int jj = 0; jj < 4; ++jj)
                r[jj] = pk_bf16(__builtin_bit_cast(f32x4, ld[kh * 8 + 2 * jj])[b], __builtin_bit_cast(f32x4, ld[kh * 8 + 2 * jj + 1])[b]);
              o[kh * 4 + b] = u32x4{r[0], r[1], r[2], r[3]};
            }
        }
        const unsigned tk = (unsigned)__builtin_amdgcn_raw_ptr_buffer_atomic_add_i32(1, rt, foff, 0, 0);
        asm volatile("" ::: "memory");
        const int dso = stream * (int)STREAM_STRIDE + ks * 8192;
        const int fso = stream * (int)FLAG_STRIDE + ks * 4;
#pragma unroll
        for (int i = 0; i < 8; ++i)
          __builtin_amdgcn_raw_buffer_store_b128(o[i], rd, lane * 16 + (i & 3) * 1024, dso + (i >> 2) * 4096, AUX_SYS);
        asm volatile("" ::: "memory");
        const unsigned tn = loc ? (unsigned)(t + NWLOC) : __builtin_amdgcn_readfirstlane(tk) + tbase;
        const bool more = tn < (unsigned)np;
        t = more ? (int)tn : t;
        DECODE_LOAD(t);
        asm volatile("s_waitcnt vmcnt(16)" ::: "memory");
        __builtin_amdgcn_raw_buffer_store_b32(seq, rf, foff, fso, AUX_SYS);
        if (!more) break;
      }
      PROF_T(1);
      return;
    }
    const unsigned seq = __builtin_amdgcn_readfirstlane(cnt) + 1u;
    {
      const char* g1 = myws + G1_OFF + (size_t)xg * FLAG_STRIDE + lane * 8;
      unsigned long long dsta = (unsigned long long)(P.ws[lane & 7] + G2_OFF +
          (size_t)((seq & 1u) * 8u + (unsigned)(lane >> 3)) * FLAG_STRIDE + (rank * 8u + (unsigned)xg) * 8u);
      asm volatile("" : "+v"(dsta));
      unsigned long long g;
      const unsigned long long t0 = __builtin_amdgcn_s_memrealtime();
      for (;;) {
        g = ld_sys64(g1);
        if (__builtin_amdgcn_ballot_w64((unsigned)(g >> 32) == seq) == ~0ull) break;
        if (wait_expired(t0, myws, seq)) break;
        __builtin_amdgcn_s_sleep(1);
      }
      const unsigned m = wave_max_u((unsigned)g);
      __hip_atomic_store((__attribute__((address_space(1))) unsigned long long*)dsta, ((unsigned long long)seq << 32) | m,
                         __ATOMIC_RELAXED, __HIP_MEMORY_SCOPE_SYSTEM);
      PROF_T(1);
    }
    if (xg < 6) {
      const char* f = myws + DONE_OFF + (size_t)xg * FLAG_STRIDE + lane * 4;
      const unsigned long long t0 = __builtin_amdgcn_s_memrealtime();
      for (;;) {
        const unsigned v = ld_sys32(f);
        if (__builtin_amdgcn_ballot_w64((int)(v - seq) >= 0) == ~0ull) break;
        if (wait_expired(t0, myws, seq)) {
          if (lane == 0) __hip_atomic_store((signed char*)(P.out[rank] + (size_t)(rank & 6u) * 512 * NLOC), (signed char)-128,
                                            __ATOMIC_RELAXED, __HIP_MEMORY_SCOPE_SYSTEM);
          break;
        }
        __builtin_amdgcn_s_sleep(1);
      }
    }
    PROF_T(2);
    return;
  }

  const unsigned cnt = __hip_atomic_load((const unsigned*)(myws + CNT_OFF), __ATOMIC_RELAXED, __HIP_MEMORY_SCOPE_AGENT);
  const int rb = 8 * (xg & 1) + (ti >> 3);
  const int cb = 8 * (xg >> 1) + (ti & 7);
  const __amdgpu_buffer_rsrc_t rA =
      __builtin_amdgcn_make_buffer_rsrc(myws + (size_t)rb * STREAM_STRIDE, 0, (int)STREAM_BYTES, RSRC_W3);
  const __amdgpu_buffer_rsrc_t rB =
      __builtin_amdgcn_make_buffer_rsrc(myws + (size_t)(16 + cb) * STREAM_STRIDE, 0, (int)STREAM_BYTES, RSRC_W3);
  const char* const fl = myws + FLAG_OFF + (size_t)(lane < 32 ? rb : 16 + cb) * FLAG_STRIDE + (lane & 31) * 4;
  const unsigned seq = __builtin_amdgcn_readfirstlane(cnt) + 1u;
  const int vo = lane * 16;

  int ready = 0;
  bool dead = false;
  f32x4 acc[4][4];
#pragma unroll
  for (int i = 0; i < 4; ++i)
#pragma unroll
    for (int j = 0; j < 4; ++j) acc[i][j] = f32x4{0.f, 0.f, 0.f, 0.f};
  u32x4 fx[5][4], fw[5][4];

#define POLL(need) \
  if (ready <= (need)) { \
    const unsigned long long t0 = __builtin_amdgcn_s_memrealtime(); \
    for (;;) { \
      const int base = ready & 32; \
      const unsigned v = ld_sys32(fl + base * 4); \
      const unsigned long long m = __builtin_amdgcn_ballot_w64((int)(v - seq) >= 0); \
      const unsigned both = (unsigned)m & (unsigned)(m >> 32); \
      const int c = (both == 0xffffffffu) ? 32 : __builtin_ctz(~both); \
      ready = base + c; \
      if (ready > (need)) break; \
      if (c == 32) continue; \
      if (wait_expired(t0, myws, seq)) { dead = true; ready = 64; break; } \
      __builtin_amdgcn_s_sleep(POLL_SLEEP); \
    } \
  } \
  asm volatile("" ::: "memory");

#define LOADH(slot, h) \
  _Pragma("unroll") for (int i = 0; i < 4; ++i) { \
    fw[slot][i] = __builtin_amdgcn_raw_buffer_load_b128(rB, vo + i * 1024, (h) * 4096, AUX_LD); \
    fx[slot][i] = __builtin_amdgcn_raw_buffer_load_b128(rA, vo + i * 1024, (h) * 4096, AUX_LD); \
  } \
  __builtin_amdgcn_sched_barrier(0);

#define MMA(slot) \
  _Pragma("unroll") for (int b = 0; b < 4; ++b) \
  _Pragma("unroll") for (int mb = 0; mb < 4; ++mb) \
    asm volatile("v_mfma_f32_16x16x32_bf16 %0, %1, %2, %0" : "+v"(acc[b][mb]) : "v"(fw[slot][b]), "v"(fx[slot][mb])); \
  __builtin_amdgcn_sched_barrier(0);

  POLL(0); LOADH(0, 0); LOADH(1, 1);
  PROF_T(5);
  POLL(1); LOADH(2, 2); LOADH(3, 3);
  for (int h = 0; h < 125; h += 5) {
#define STEP(u) { const int hn = (h + u + 4 < 127) ? h + u + 4 : 127; POLL(hn >> 1); LOADH((u + 4) % 5, hn); MMA(u); }
    STEP(0) STEP(1) STEP(2) STEP(3) STEP(4)
  }
  MMA(0) MMA(1) MMA(2)
  asm volatile("s_nop 7\n\ts_nop 7\n\ts_nop 7" ::: "memory");

  float mx = 0.f;
#pragma unroll
  for (int b = 0; b < 4; ++b)
#pragma unroll
    for (int mb = 0; mb < 4; ++mb)
#pragma unroll
      for (int r = 0; r < 4; ++r) mx = fmaxf(mx, acc[b][mb][r]);
  const unsigned mxu = wave_max_u(__builtin_bit_cast(unsigned, mx));
  if (lane == 0)
    st_sys64(myws + G1_OFF + (size_t)xg * FLAG_STRIDE + ti * 8, ((unsigned long long)seq << 32) | mxu);

  PROF_T(1);
  const unsigned pr = rank >> 1;
  unsigned owner, col0, row0, dslot = 0;
  if (cb < 8) { owner = rank; col0 = 64 * cb; }
  else {
    const unsigned u = (cb - 8) >> 2, c = (cb - 8) & 3, j = u >> 1;
    owner = (((pr ^ (j + 1)) << 1) | (u & 1)) & 7u;
    col0 = 256 * ((rank ^ owner) & 1) + 64 * c;
    dslot = (2 * j + (rank & 1)) * FLAG_STRIDE + (rb * 4 + c) * 4;
  }
  if (rb < 10) row0 = 512 * (2 * pr + (rb >= 5)) + 64 * (rb >= 5 ? rb - 5 : rb);
  else {
    const unsigned u = rb - 10, j = u >> 1;
    row0 = 512 * ((((pr ^ (j + 1)) << 1) | (u & 1)) & 7u) + 320 + 64 * j;
  }
  const __amdgpu_buffer_rsrc_t rO = __builtin_amdgcn_make_buffer_rsrc(P.out[owner], 0, KDIM * NLOC, RSRC_W3);

  float gm;
  {
    const char* g2 = myws + G2_OFF + (size_t)((seq & 1u) * 8u + (unsigned)xg) * FLAG_STRIDE + lane * 8;
    const unsigned long long t0 = __builtin_amdgcn_s_memrealtime();
    unsigned long long g;
    unsigned tv;
    for (;;) {
      g = ld_sys64(g2);
      tv = __hip_atomic_load((const unsigned*)(myws + TMO_OFF), __ATOMIC_RELAXED, __HIP_MEMORY_SCOPE_AGENT);
      if (__builtin_amdgcn_ballot_w64((unsigned)(g >> 32) == seq) == ~0ull) break;
      if (wait_expired(t0, myws, seq)) { dead = true; break; }
      __builtin_amdgcn_s_sleep(1);
    }
    if (__builtin_amdgcn_readfirstlane(tv) == seq) dead = true;
    gm = __builtin_bit_cast(float, wave_max_u((unsigned)g));
  }
  PROF_T(2);
  if (xg == 0 && ti == 0 && lane == 0)
    __hip_atomic_store((unsigned*)(myws + CNT_OFF), seq, __ATOMIC_RELAXED, __HIP_MEMORY_SCOPE_AGENT);
  const float inv = gm > 0.f ? 127.0f / gm : 0.f;
#pragma unroll
  for (int mb = 0; mb < 4; ++mb) {
    u32x4 o;
#pragma unroll
    for (int r = 0; r < 4; ++r) {
      unsigned wd = 0;
#pragma unroll
      for (int b = 0; b < 4; ++b) {
        const float v = fminf(fmaxf(__builtin_rintf(acc[b][mb][r] * inv), 0.f), 127.f);
        wd |= (unsigned)v << (8 * b);
      }
      o[r] = dead ? 0x80808080u : wd;
    }
    __builtin_amdgcn_raw_buffer_store_b128(o, rO, (int)((row0 + l15) * NLOC + col0 + 16 * q), mb * 16 * NLOC, AUX_SYS);
  }
  if (cb >= 8) {
    asm volatile("s_waitcnt vmcnt(0)" ::: "memory");
    if (lane == 0) st_sys32(P.ws[owner] + DONE_OFF + dslot, seq);
  }
  PROF_T(3);
}

extern "C" void kernel_launch(int rank, int nranks, ncclComm_t comm, hipStream_t stream,
                              void* const* d_in, const int* in_sizes, int n_in,
                              void* d_out, int out_size,
                              const int* in_dtypes, int out_dtype,
                              void* const* d_out_all,
                              void* const* d_ws_all, size_t ws_bytes) {
  (void)comm; (void)in_dtypes; (void)out_dtype; (void)d_out;
  if (nranks != NR || (unsigned)rank >= (unsigned)NR || n_in < 2 || ws_bytes < WS_NEED ||
      in_sizes[0] != 512 * KDIM || in_sizes[1] != KDIM * NLOC || out_size != KDIM * NLOC) {
    fprintf(stderr, "unexpected problem shape\n");
    return;
  }
  Params p;
  p.x = (const float*)d_in[0];
  p.w = (const float*)d_in[1];
  for (int i = 0; i < NR; ++i) { p.ws[i] = (char*)d_ws_all[i]; p.out[i] = (char*)d_out_all[i]; }
  p.rank = rank;
  ag_gemm_fused<<<256, 256, LDS_PAD, stream>>>(p);
  HIP_CHECK(hipGetLastError());
}
```

```cpp
#include <hip/hip_runtime.h>
#include <rccl/rccl.h>
#include <cstdio>

#define HIP_CHECK(expr) do { hipError_t _e = (expr); if (_e != hipSuccess) { \
    fprintf(stderr, "HIP error %s at %s:%d\n", hipGetErrorString(_e), __FILE__, __LINE__); \
    return; } } while (0)

namespace {

typedef __attribute__((ext_vector_type(4))) unsigned u32x4;
typedef __attribute__((ext_vector_type(2))) unsigned u32x2;
typedef __attribute__((ext_vector_type(4))) float f32x4;

#define NWL 5
#define NSEND (7 * NWL)
#define NPSH 49
#define NPUSH 1
#define T0 (7 * NPUSH)
#define LOC_OWN 0
#define AVS 4
#define PEELP 0
#define GSLEEP 0
#define KROT 1
#define CSLEEP 60
#define NWLOC 176
#define NSB 10
#define GPW 3
#define LDS_PAD (84 * 1024)
#define AUX_LD 17
#define POLL_SLEEP 4

constexpr int NR = 8;
constexpr int KDIM = 4096, NLOC = 512;
constexpr unsigned STREAM_BYTES = 512u * 1024u;
constexpr unsigned STREAM_STRIDE = STREAM_BYTES + 4096u;
constexpr unsigned NSTREAM = 51;
constexpr unsigned FLAG_STRIDE = 4352;
constexpr size_t CTRL_OFF = 26ull << 20;
constexpr size_t CNT_OFF = CTRL_OFF;
constexpr size_t TMO_OFF = CTRL_OFF + 256;
constexpr size_t TICK_OFF = CTRL_OFF + 1024;
constexpr size_t G1_OFF = CTRL_OFF + 8192;
constexpr size_t G2_OFF = CTRL_OFF + 65536;
constexpr size_t DONE_OFF = CTRL_OFF + 196608;
constexpr size_t AV_OFF = CTRL_OFF + 262144;
constexpr int AV_LINK = 336 * AVS;
constexpr size_t WS_NEED = AV_OFF + 8 * AV_LINK;
static_assert((size_t)NSTREAM * STREAM_STRIDE <= CTRL_OFF, "layout");
constexpr int AUX_SYS = 17;
constexpr int RSRC_W3 = 0x00020000;
constexpr unsigned long long DEADLINE_TICKS = 100000000ull;
constexpr unsigned long long TMO_LOOK_TICKS = 20000ull;

#define PROF_T(slot) do { } while (0)
#define PROF_V(slot, v) do { } while (0)
#define PROF_S(x)

struct Params {
  const float* x;
  const float* w;
  char* myws;
  unsigned rank, pad;
  char* ws[NR];
  char* out[NR];
};

constexpr unsigned long long src_table(int part) {
  unsigned long long t = 0;
  for (int i = 0; i < 16; ++i) {
    const int u = 16 * part + i;
    const int sub = u >= 35 ? u - 35 : u / 7, d = u >= 35 ? 0 : u % 7 + 1;
    const bool ux = d <= 1 ? sub < 5 : sub == 0;
    const int b = d <= 1 ? (ux ? sub : sub - 5) : (ux ? 4 + (d >> 1) : 4 * (d & 1) + sub - 1);
    t |= (unsigned long long)((ux ? 8 : 0) | b) << (4 * i);
  }
  return t;
}

__device__ __forceinline__ unsigned pk_bf16(float a, float b) {
  unsigned r;
  asm("v_cvt_pk_bf16_f32 %0, %1, %2" : "=v"(r) : "v"(a), "v"(b));
  return r;
}
__device__ __forceinline__ unsigned ld_sys32(const void* p) {
  return __hip_atomic_load((const unsigned*)p, __ATOMIC_RELAXED, __HIP_MEMORY_SCOPE_SYSTEM);
}
__device__ __forceinline__ unsigned long long ld_sys64(const void* p) {
  return __hip_atomic_load((const unsigned long long*)p, __ATOMIC_RELAXED, __HIP_MEMORY_SCOPE_SYSTEM);
}
__device__ __forceinline__ void st_sys32(void* p, unsigned v) {
  __hip_atomic_store((unsigned*)p, v, __ATOMIC_RELAXED, __HIP_MEMORY_SCOPE_SYSTEM);
}
__device__ __forceinline__ void st_sys64(void* p, unsigned long long v) {
  __hip_atomic_store((unsigned long long*)p, v, __ATOMIC_RELAXED, __HIP_MEMORY_SCOPE_SYSTEM);
}

__device__ __forceinline__ bool wait_expired(unsigned long long t0, char* myws, unsigned seq) {
  const unsigned long long el = __builtin_amdgcn_s_memrealtime() - t0;
  if (el > TMO_LOOK_TICKS) {
    unsigned* tmo = (unsigned*)(myws + TMO_OFF);
    const unsigned tv = __builtin_amdgcn_readfirstlane(
        __hip_atomic_load(tmo, __ATOMIC_RELAXED, __HIP_MEMORY_SCOPE_AGENT));
    if (tv == seq || el > DEADLINE_TICKS) {
      __hip_atomic_store(tmo, seq, __ATOMIC_RELAXED, __HIP_MEMORY_SCOPE_AGENT);
      return true;
    }
  }
  return false;
}

__device__ __forceinline__ unsigned wave_max_u(unsigned v) {
  v = max(v, (unsigned)__builtin_amdgcn_update_dpp(0, (int)v, 0x111, 0xf, 0xf, false));
  v = max(v, (unsigned)__builtin_amdgcn_update_dpp(0, (int)v, 0x112, 0xf, 0xf, false));
  v = max(v, (unsigned)__builtin_amdgcn_update_dpp(0, (int)v, 0x114, 0xf, 0xf, false));
  v = max(v, (unsigned)__builtin_amdgcn_update_dpp(0, (int)v, 0x118, 0xf, 0xf, false));
  v = max(v, (unsigned)__builtin_amdgcn_update_dpp(0, (int)v, 0x142, 0xa, 0xf, false));
  v = max(v, (unsigned)__builtin_amdgcn_update_dpp(0, (int)v, 0x143, 0xc, 0xf, false));
  return (unsigned)__builtin_amdgcn_readlane((int)v, 63);
}

}

__global__ void __launch_bounds__(256, 2) ag_gemm_fused(Params P) {
  extern __shared__ __attribute__((aligned(16))) char lds[];
  const int lane = threadIdx.x & 63;
  const int wv = __builtin_amdgcn_readfirstlane(threadIdx.x >> 6);
  const int blk = blockIdx.x;
  const unsigned rank = P.rank & 7u;
  char* const myws = P.myws;
  const float* px = P.x;
  const float* pw = P.w;
  asm volatile("" : "+s"(px), "+s"(pw));
  const int l15 = lane & 15, q = lane >> 4;
  PROF_T(0);

  const int xg = blk & 7, li = blk >> 3;
  const int ti = (li - NSB) * GPW + wv;
  if (__builtin_expect(li < NSB || wv >= GPW || ti >= 64, 1)) {
    int sw;
    if (li < NSB) {
      if (wv == 0) { sw = blk; if (sw >= NPSH) return; }
      else { sw = -2; if (wv != 1 || blk >= NSEND) return; }
    }
    else if (wv >= GPW) { sw = NPSH + (blk - 8 * NSB); if (wv > GPW || sw >= NPSH + NWLOC) return; }
    else { if (ti != 64) return; sw = -1; }
    if (__builtin_expect(sw >= 0, 1)) {
      const bool loc = sw >= NPSH;
      const int np = loc ? 16 * 64 : NPSH * NPUSH, magic = loc ? 4096 : 1873, um = loc ? 16 : 35, gs = loc ? NWLOC : NPSH;
      const int voffx = (q * KDIM + 4 * l15) * 4, voffw = ((4 * q) * NLOC + 4 * l15) * 4;
      char* const lw = lds + wv * 10240 + (l15 >> 3) * 5120 + (l15 & 3) * 320 + q * 16 + ((l15 >> 2) & 1) * 8;
      const char* const lr = lds + wv * 10240 + q * 320 + l15 * 16;
      bool isx;
      int ks, un;
      u32x4 ld[16], o[8];
#define DECODE_LOAD(g, valid) { \
        ks = ((g) * magic) >> 16; \
        un = (g) - ks * um; \
        if (loc && ks < KROT + 4) ks = ks < 4 ? ks + KROT : ks - 4; \
        const unsigned ts = (unsigned)((loc ? 0x76543210fedcba98ull : un < 16 ? src_table(0) : un < 32 ? src_table(1) : src_table(2)) >> (4 * (un & 15))); \
        isx = (ts & 8u) != 0; \
        const __amdgpu_buffer_rsrc_t rs = __builtin_amdgcn_make_buffer_rsrc((void*)(isx ? px : pw), 0, (valid) ? 512 * 4096 * 4 : 0, RSRC_W3); \
        const int vo_ = isx ? voffx : voffw; \
        const int so_ = (int)(ts & 7u) * (isx ? 64 * KDIM * 4 : 256) + ks * (isx ? 256 : 64 * NLOC * 4); \
        const int sa_ = isx ? 4 * KDIM * 4 : NLOC * 4, sc_ = isx ? 16 * KDIM * 4 : 16 * NLOC * 4; \
        _Pragma("unroll") for (int i = 0; i < 16; ++i) \
          ld[i] = __builtin_amdgcn_raw_buffer_load_b128(rs, vo_, so_ + (i & 3) * sa_ + (i >> 2) * sc_, 0); \
        asm volatile("" ::: "memory"); \
      }
#define CONVERT() { \
        if (isx) { \
          _Pragma("unroll") for (int i = 0; i < 16; ++i) { \
            const f32x4 a = __builtin_bit_cast(f32x4, ld[i]); \
            *(u32x2*)(lw + (i >> 2) * 1280 + (i & 3) * 64) = u32x2{pk_bf16(a.x, a.y), pk_bf16(a.z, a.w)}; \
          } \
          _Pragma("unroll") for (int i = 0; i < 8; ++i) o[i] = *(const u32x4*)(lr + i * 1280); \
        } else { \
          _Pragma("unroll") for (int kh = 0; kh < 2; ++kh) \
          _Pragma("unroll") for (int b = 0; b < 4; ++b) { \
            unsigned r[4]; \
            _Pragma("unroll") for (int jj = 0; jj < 4; ++jj) \
              r[jj] = pk_bf16(__builtin_bit_cast(f32x4, ld[kh * 8 + 2 * jj])[b], __builtin_bit_cast(f32x4, ld[kh * 8 + 2 * jj + 1])[b]); \
            o[kh * 4 + b] = u32x4{r[0], r[1], r[2], r[3]}; \
          } \
        } \
        dks = ks; dun = un; \
        dst = (char*)(((unsigned long long)(unsigned)__builtin_amdgcn_readlane((int)(tdst >> 32), un) << 32) | \
                      (unsigned)__builtin_amdgcn_readlane((int)tdst, un)); \
        asm volatile("" ::: "memory"); \
      }
      int g = loc ? sw - NPSH : sw, dks, dun, fks = 0, fun = -2;
      char* dst;
      if (CSLEEP && sw >= NPSH + 64) __builtin_amdgcn_s_sleep(CSLEEP);
      DECODE_LOAD(g, true);
      __builtin_amdgcn_sched_barrier(0);
      const unsigned cnt = __hip_atomic_load((const unsigned*)(myws + CNT_OFF), __ATOMIC_RELAXED, __HIP_MEMORY_SCOPE_AGENT);
      unsigned long long tdst;
      char* tflag;
      int tkey, tstep = 4;
      {
        const int r1 = (int)(rank & 1);
        if (loc) {
          const int e = lane >> 2, i = lane & 3;
          const int d = i == 0 ? 0 : e < 5 ? 1 : e < 8 ? 2 * e + i - 9 : 2 * i + ((e >> 2) & 1);
          const int sub = e < 5 ? e : e < 8 ? 0 : (e & 3) + 1;
          const bool ok = e < 5 ? i < 2 : e < 8 ? (i == 1 || i == 2) : true;
          tkey = ok ? e : -1;
          tstep = i == 0 ? 4 : 5 * AVS;
          const unsigned off = i == 0 ? (unsigned)(e < 5 ? 5 * r1 + e : e < 8 ? 43 + e : 8 + e) * STREAM_STRIDE + STREAM_BYTES
                                      : (unsigned)AV_OFF + (unsigned)(((d - 1) & 7) * AV_LINK + sub * AVS);
          tflag = P.ws[(rank ^ (unsigned)d) & 7u] + off;
          tdst = (unsigned long long)myws + (unsigned)(l15 < 5 ? 5 * r1 + l15 : l15 < 8 ? 43 + l15 : 8 + l15) * STREAM_STRIDE;
        } else {
          const int u = lane < 35 ? lane : 34;
          const int sub = (u * 37) >> 8, d = u - 7 * sub + 1, j = (d >> 1) - 1;
          const int stream = d == 1 ? 5 * r1 + sub : (sub == 0 ? 10 + 2 * j + r1 : 23 + 4 * (2 * j + r1) + sub);
          tkey = lane;
          tdst = (unsigned long long)P.ws[(rank ^ (unsigned)d) & 7u] + (unsigned)stream * STREAM_STRIDE;
          tflag = (char*)tdst + STREAM_BYTES;
        }
      }
      CONVERT();
      const unsigned seq = __builtin_amdgcn_readfirstlane(cnt) + 1u;
#define STORES() { \
        const __amdgpu_buffer_rsrc_t rd = __builtin_amdgcn_make_buffer_rsrc(dst, 0, (int)STREAM_BYTES, RSRC_W3); \
        _Pragma("unroll") for (int i = 0; i < 8; ++i) \
          __builtin_amdgcn_raw_buffer_store_b128(o[i], rd, lane * 16 + (i & 3) * 1024, dks * 8192 + (i >> 2) * 4096, AUX_SYS); \
        asm volatile("" ::: "memory"); \
      }
      if (loc || PEELP) {
        STORES();
        g += gs;
        DECODE_LOAD(g, true);
        asm volatile("s_waitcnt vmcnt(16)" ::: "memory");
        if (tkey == dun) st_sys32(tflag + dks * tstep, seq);
        PROF_T(2);
        CONVERT();
      }
      bool more = g + gs < np;
      g = more ? g + gs : g;
      DECODE_LOAD(g, more);
      for (;;) {
        STORES();
        asm volatile("s_waitcnt vmcnt(24)" ::: "memory");
        if (tkey == fun) st_sys32(tflag + fks * tstep, seq);
        asm volatile("" ::: "memory");
        fun = dun; fks = dks;
        if (!more) break;
        CONVERT();
        more = g + gs < np;
        g = more ? g + gs : g;
        DECODE_LOAD(g, more);
      }
      asm volatile("s_waitcnt vmcnt(0)" ::: "memory");
      if (tkey == fun) st_sys32(tflag + fks * tstep, seq);
      PROF_T(1);
      return;
    }
    if (sw == -2) {
      const unsigned cnt = __hip_atomic_load((const unsigned*)(myws + CNT_OFF), __ATOMIC_RELAXED, __HIP_MEMORY_SCOPE_AGENT);
      const int w = (blk * 37) >> 8, dl = blk - 7 * w, d = dl + 1, j = (d >> 1) - 1;
      const unsigned sr = (rank ^ (unsigned)d) & 7u;
      const int s1 = (int)(sr & 1);
      char* const sws = P.ws[sr];
      const int src0 = d == 1 ? 5 * s1 : 48 + j, srcn = d == 1 ? 5 * s1 : 15 + 4 * (d & 1);
      const int dst0 = d == 1 ? 5 * s1 : 10 + 2 * j + s1, dstn = d == 1 ? 5 * s1 : 23 + 4 * (2 * j + s1);
      const int foff = lane * (1 << 20);
      const __amdgpu_buffer_rsrc_t rt = __builtin_amdgcn_make_buffer_rsrc(myws + TICK_OFF + dl * 256, 0, 4, RSRC_W3);
      const char* const av = myws + AV_OFF + dl * AV_LINK + min(lane, 15) * AVS;
      const unsigned seq = __builtin_amdgcn_readfirstlane(cnt) + 1u;
      const unsigned tbase = (unsigned)(T0 + 2 * NWL) - (seq - 1u) * (unsigned)(320 - T0 - NWL);
      u32x4 b0[8], b1[8];
      bool dead = false;
#define AVAIL(t, mask) { \
        const unsigned long long t0 = __builtin_amdgcn_s_memrealtime(); \
        while (!dead) { \
          const unsigned v = ld_sys32(av + (t) * AVS); \
          if (((unsigned)__builtin_amdgcn_ballot_w64((int)(v - seq) >= 0) & (mask)) == (mask)) break; \
          dead = wait_expired(t0, myws, seq); \
          __builtin_amdgcn_s_sleep(1); \
        } \
        asm volatile("" ::: "memory"); \
      }
#define PULL(buf, t, valid) { \
        const int ks_ = ((t) * 13108) >> 16, sub_ = (t) - 5 * ks_; \
        const __amdgpu_buffer_rsrc_t rs = __builtin_amdgcn_make_buffer_rsrc( \
            sws + (unsigned)(sub_ == 0 ? src0 : srcn + sub_) * STREAM_STRIDE, 0, (valid) ? (int)STREAM_BYTES : 0, RSRC_W3); \
        _Pragma("unroll") for (int i = 0; i < 8; ++i) \
          buf[i] = __builtin_amdgcn_raw_buffer_load_b128(rs, lane * 16 + (i & 3) * 1024, ks_ * 8192 + (i >> 2) * 4096, AUX_SYS); \
        asm volatile("" ::: "memory"); \
      }
      int tc = T0 + w, tn = T0 + NWL + w, fsz = 0;
      char* fdst = myws;
      bool vn = true;
      AVAIL(tc, 1u | (1u << NWL));
      PROF_S(unsigned long long tB = __builtin_amdgcn_s_memrealtime(); )
      PULL(b0, tc, true);
      unsigned tk = (unsigned)__builtin_amdgcn_raw_ptr_buffer_atomic_add_i32(1, rt, foff, 0, 0);
      unsigned avv = ld_sys32(av + tn * AVS);
      PULL(b1, tn, true);
#define PSTEP(buf) { \
        asm volatile("s_waitcnt vmcnt(8)" ::: "memory"); \
        __builtin_amdgcn_raw_buffer_store_b32(seq, __builtin_amdgcn_make_buffer_rsrc(fdst, 0, fsz, RSRC_W3), foff, 0, AUX_SYS); \
        PROF_S(__builtin_amdgcn_raw_buffer_store_b32((unsigned)__builtin_amdgcn_s_memrealtime(), __builtin_amdgcn_make_buffer_rsrc(myws + (41ull << 20) + (dl * 320 + tc) * 4, 0, 4, RSRC_W3), foff, 0, 0);) \
        const int ks_ = (tc * 13108) >> 16, sub_ = tc - 5 * ks_; \
        char* const dst = myws + (unsigned)(sub_ == 0 ? dst0 : dstn + sub_) * STREAM_STRIDE; \
        const __amdgpu_buffer_rsrc_t rd = __builtin_amdgcn_make_buffer_rsrc(dst, 0, (int)STREAM_BYTES, RSRC_W3); \
        _Pragma("unroll") for (int i = 0; i < 8; ++i) \
          __builtin_amdgcn_raw_buffer_store_b128(buf[i], rd, lane * 16 + (i & 3) * 1024, ks_ * 8192 + (i >> 2) * 4096, AUX_SYS); \
        asm volatile("" ::: "memory"); \
        fdst = dst + STREAM_BYTES + ks_ * 4; fsz = 4; \
        if (!vn) break; \
        const unsigned t2u = __builtin_amdgcn_readfirstlane(tk) + tbase; \
        vn = t2u < 320u; \
        const int t2 = vn ? (int)t2u : 319; \
        const unsigned long long m = __builtin_amdgcn_ballot_w64((int)(avv - seq) >= 0); \
        const unsigned rel = (unsigned)(t2 - tn); \
        if (vn && !(rel < 16u && ((m >> (rel & 15u)) & 1ull))) AVAIL(t2, 1u); \
        tk = (unsigned)__builtin_amdgcn_raw_ptr_buffer_atomic_add_i32(1, rt, vn ? foff : foff + (1 << 20), 0, 0); \
        avv = ld_sys32(av + t2 * AVS); \
        PULL(buf, t2, vn); \
        tc = tn; tn = t2; \
      }
      for (;;) { PSTEP(b0) PSTEP(b1) }
      asm volatile("s_waitcnt vmcnt(0)" ::: "memory");
      __builtin_amdgcn_raw_buffer_store_b32(seq, __builtin_amdgcn_make_buffer_rsrc(fdst, 0, 4, RSRC_W3), foff, 0, AUX_SYS);
      PROF_T(1);
      PROF_V(2, tB);
      return;
    }
    const unsigned cnt = __hip_atomic_load((const unsigned*)(myws + CNT_OFF), __ATOMIC_RELAXED, __HIP_MEMORY_SCOPE_AGENT);
    const unsigned seq = __builtin_amdgcn_readfirstlane(cnt) + 1u;
    {
      const char* g1 = myws + G1_OFF + (size_t)xg * FLAG_STRIDE + lane * 8;
      unsigned long long dsta = (unsigned long long)(P.ws[lane & 7] + G2_OFF +
          (size_t)((seq & 1u) * 8u + (unsigned)(lane >> 3)) * FLAG_STRIDE + (rank * 8u + (unsigned)xg) * 8u);
      asm volatile("" : "+v"(dsta));
      unsigned long long g;
      const unsigned long long t0 = __builtin_amdgcn_s_memrealtime();
      for (;;) {
        g = ld_sys64(g1);
        if (__builtin_amdgcn_ballot_w64((unsigned)(g >> 32) == seq) == ~0ull) break;
        if (wait_expired(t0, myws, seq)) break;
        __builtin_amdgcn_s_sleep(1);
      }
      const unsigned m = wave_max_u((unsigned)g);
      __hip_atomic_store((__attribute__((address_space(1))) unsigned long long*)dsta, ((unsigned long long)seq << 32) | m,
                         __ATOMIC_RELAXED, __HIP_MEMORY_SCOPE_SYSTEM);
      PROF_T(1);
    }
    if (xg < 6) {
      const char* f = myws + DONE_OFF + (size_t)xg * FLAG_STRIDE + lane * 4;
      const unsigned long long t0 = __builtin_amdgcn_s_memrealtime();
      for (;;) {
        const unsigned v = ld_sys32(f);
        if (__builtin_amdgcn_ballot_w64((int)(v - seq) >= 0) == ~0ull) break;
        if (wait_expired(t0, myws, seq)) {
          if (lane == 0) __hip_atomic_store((signed char*)(P.out[rank] + (size_t)(rank & 6u) * 512 * NLOC), (signed char)-128,
                                            __ATOMIC_RELAXED, __HIP_MEMORY_SCOPE_SYSTEM);
          break;
        }
        __builtin_amdgcn_s_sleep(1);
      }
    }
    PROF_T(2);
    return;
  }

  const unsigned cnt = __hip_atomic_load((const unsigned*)(myws + CNT_OFF), __ATOMIC_RELAXED, __HIP_MEMORY_SCOPE_AGENT);
  const int rb = 8 * (xg & 1) + (ti >> 3);
  const int cb = 8 * (xg >> 1) + (ti & 7);
  const __amdgpu_buffer_rsrc_t rA =
      __builtin_amdgcn_make_buffer_rsrc(myws + (size_t)rb * STREAM_STRIDE, 0, (int)STREAM_BYTES, RSRC_W3);
  const __amdgpu_buffer_rsrc_t rB =
      __builtin_amdgcn_make_buffer_rsrc(myws + (size_t)(16 + cb) * STREAM_STRIDE, 0, (int)STREAM_BYTES, RSRC_W3);
  const char* const fl = myws + (size_t)(lane < 32 ? rb : 16 + cb) * STREAM_STRIDE + STREAM_BYTES + (lane & 31) * 4;
  const unsigned seq = __builtin_amdgcn_readfirstlane(cnt) + 1u;
  const int vo = lane * 16;

  int ready = 0;
  bool dead = false;
  f32x4 acc[4][4];
#pragma unroll
  for (int i = 0; i < 4; ++i)
#pragma unroll
    for (int j = 0; j < 4; ++j) acc[i][j] = f32x4{0.f, 0.f, 0.f, 0.f};
  u32x4 fx[5][4], fw[5][4];

#define POLL(need) \
  if (ready <= (need)) { \
    const unsigned long long t0 = __builtin_amdgcn_s_memrealtime(); \
    for (;;) { \
      const int base = ready & 32; \
      const unsigned v = ld_sys32(fl + base * 4); \
      const unsigned long long m = __builtin_amdgcn_ballot_w64((int)(v - seq) >= 0); \
      const unsigned both = (unsigned)m & (unsigned)(m >> 32); \
      const int c = (both == 0xffffffffu) ? 32 : __builtin_ctz(~both); \
      ready = base + c; \
      if (ready > (need)) break; \
      if (c == 32) continue; \
      if (wait_expired(t0, myws, seq)) { dead = true; ready = 64; break; } \
      __builtin_amdgcn_s_sleep(POLL_SLEEP); \
    } \
  } \
  asm volatile("" ::: "memory");

#define LOADH(slot, h) \
  _Pragma("unroll") for (int i = 0; i < 4; ++i) { \
    fw[slot][i] = __builtin_amdgcn_raw_buffer_load_b128(rB, vo + i * 1024, (h) * 4096, AUX_LD); \
    fx[slot][i] = __builtin_amdgcn_raw_buffer_load_b128(rA, vo + i * 1024, (h) * 4096, AUX_LD); \
  } \
  __builtin_amdgcn_sched_barrier(0);

#define MMA(slot) \
  _Pragma("unroll") for (int b = 0; b < 4; ++b) \
  _Pragma("unroll") for (int mb = 0; mb < 4; ++mb) \
    asm volatile("v_mfma_f32_16x16x32_bf16 %0, %1, %2, %0" : "+v"(acc[b][mb]) : "v"(fw[slot][b]), "v"(fx[slot][mb])); \
  __builtin_amdgcn_sched_barrier(0);

  if (GSLEEP) __builtin_amdgcn_s_sleep(GSLEEP);
  POLL(0); LOADH(0, 0); LOADH(1, 1);
  PROF_T(5);
  POLL(1); LOADH(2, 2); LOADH(3, 3);
  for (int h = 0; h < 125; h += 5) {
#define STEP(u) { const int hn = (h + u + 4 < 127) ? h + u + 4 : 127; POLL(hn >> 1); LOADH((u + 4) % 5, hn); MMA(u); }
    STEP(0) STEP(1) STEP(2) STEP(3) STEP(4)
  }
  MMA(0) MMA(1) MMA(2)
  asm volatile("s_nop 7\n\ts_nop 7\n\ts_nop 7" ::: "memory");

  float mx = 0.f;
#pragma unroll
  for (int b = 0; b < 4; ++b)
#pragma unroll
    for (int mb = 0; mb < 4; ++mb)
#pragma unroll
      for (int r = 0; r < 4; ++r) mx = fmaxf(mx, acc[b][mb][r]);
  const unsigned mxu = wave_max_u(__builtin_bit_cast(unsigned, mx));
  if (lane == 0)
    st_sys64(myws + G1_OFF + (size_t)xg * FLAG_STRIDE + ti * 8, ((unsigned long long)seq << 32) | mxu);

  PROF_T(1);
  const unsigned pr = rank >> 1;
  unsigned owner, col0, row0, dslot = 0;
  if (cb < 8) { owner = rank; col0 = 64 * cb; }
  else {
    const unsigned u = (cb - 8) >> 2, c = (cb - 8) & 3, j = u >> 1;
    owner = (((pr ^ (j + 1)) << 1) | (u & 1)) & 7u;
    col0 = 256 * ((rank ^ owner) & 1) + 64 * c;
    dslot = (2 * j + (rank & 1)) * FLAG_STRIDE + (rb * 4 + c) * 4;
  }
  if (rb < 10) row0 = 512 * (2 * pr + (rb >= 5)) + 64 * (rb >= 5 ? rb - 5 : rb);
  else {
    const unsigned u = rb - 10, j = u >> 1;
    row0 = 512 * ((((pr ^ (j + 1)) << 1) | (u & 1)) & 7u) + 320 + 64 * j;
  }
  const __amdgpu_buffer_rsrc_t rO = __builtin_amdgcn_make_buffer_rsrc(P.out[owner], 0, KDIM * NLOC, RSRC_W3);

  float gm;
  {
    const char* g2 = myws + G2_OFF + (size_t)((seq & 1u) * 8u + (unsigned)xg) * FLAG_STRIDE + lane * 8;
    const unsigned long long t0 = __builtin_amdgcn_s_memrealtime();
    unsigned long long g;
    unsigned tv;
    for (;;) {
      g = ld_sys64(g2);
      tv = __hip_atomic_load((const unsigned*)(myws + TMO_OFF), __ATOMIC_RELAXED, __HIP_MEMORY_SCOPE_AGENT);
      if (__builtin_amdgcn_ballot_w64((unsigned)(g >> 32) == seq) == ~0ull) break;
      if (wait_expired(t0, myws, seq)) { dead = true; break; }
      __builtin_amdgcn_s_sleep(1);
    }
    if (__builtin_amdgcn_readfirstlane(tv) == seq) dead = true;
    gm = __builtin_bit_cast(float, wave_max_u((unsigned)g));
  }
  PROF_T(2);
  if (xg == 0 && ti == 0 && lane == 0)
    __hip_atomic_store((unsigned*)(myws + CNT_OFF), seq, __ATOMIC_RELAXED, __HIP_MEMORY_SCOPE_AGENT);
  const float inv = gm > 0.f ? 127.0f / gm : 0.f;
#pragma unroll
  for (int mb = 0; mb < 4; ++mb) {
    u32x4 o;
#pragma unroll
    for (int r = 0; r < 4; ++r) {
      unsigned wd = 0;
#pragma unroll
      for (int b = 0; b < 4; ++b) {
        const float v = fminf(fmaxf(__builtin_rintf(acc[b][mb][r] * inv), 0.f), 127.f);
        wd |= (unsigned)v << (8 * b);
      }
      o[r] = dead ? 0x80808080u : wd;
    }
    __builtin_amdgcn_raw_buffer_store_b128(o, rO, (int)((row0 + l15) * NLOC + col0 + 16 * q), mb * 16 * NLOC, AUX_SYS);
  }
  if (cb >= 8) {
    asm volatile("s_waitcnt vmcnt(0)" ::: "memory");
    if (lane == 0) st_sys32(P.ws[owner] + DONE_OFF + dslot, seq);
  }
  PROF_T(3);
}

extern "C" void kernel_launch(int rank, int nranks, ncclComm_t comm, hipStream_t stream,
                              void* const* d_in, const int* in_sizes, int n_in,
                              void* d_out, int out_size,
                              const int* in_dtypes, int out_dtype,
                              void* const* d_out_all,
                              void* const* d_ws_all, size_t ws_bytes) {
  (void)comm; (void)in_dtypes; (void)out_dtype; (void)d_out;
  if (nranks != NR || (unsigned)rank >= (unsigned)NR || n_in < 2 || ws_bytes < WS_NEED ||
      in_sizes[0] != 512 * KDIM || in_sizes[1] != KDIM * NLOC || out_size != KDIM * NLOC) {
    fprintf(stderr, "unexpected problem shape\n");
    return;
  }
  Params p;
  p.x = (const float*)d_in[0];
  p.w = (const float*)d_in[1];
  for (int i = 0; i < NR; ++i) { p.ws[i] = (char*)d_ws_all[i]; p.out[i] = (char*)d_out_all[i]; }
  p.myws = p.ws[rank];
  p.rank = (unsigned)rank;
  p.pad = 0;
  ag_gemm_fused<<<256, 256, LDS_PAD, stream>>>(p);
  HIP_CHECK(hipGetLastError());
}
```

```cpp
#include <hip/hip_runtime.h>
#include <rccl/rccl.h>
#include <cstdio>

#define HIP_CHECK(expr) do { hipError_t _e = (expr); if (_e != hipSuccess) { \
    fprintf(stderr, "HIP error %s at %s:%d\n", hipGetErrorString(_e), __FILE__, __LINE__); \
    return; } } while (0)

namespace {

typedef __attribute__((ext_vector_type(4))) unsigned u32x4;
typedef __attribute__((ext_vector_type(2))) unsigned u32x2;
typedef __attribute__((ext_vector_type(4))) float f32x4;

#define NWL 5
#define NSEND (7 * NWL)
#define NPSH 56
#define NPUSH 1
#define T0 (NPSH / 7 * NPUSH)
#define LOC_OWN 0
#define AVS 4
#define KROT 1
#define CSLEEP 60
#define NWLOC 176
#define NSB 10
#define GPW 3
#define LDS_PAD (84 * 1024)
#define AUX_LD 17
#define G2SLEEP 3
#define G2LONG 24
#define G2CNT 1
#define CLONG 12
#define DLONG 8
#define PSLEEP 18
#define PEARLY 3
#define POLL_SLEEP 4

constexpr int NR = 8;
constexpr int KDIM = 4096, NLOC = 512;
constexpr unsigned STREAM_BYTES = 512u * 1024u;
constexpr unsigned STREAM_STRIDE = STREAM_BYTES + 4096u;
constexpr unsigned NSTREAM = 51;
constexpr unsigned FLAG_STRIDE = 4352;
constexpr size_t CTRL_OFF = 26ull << 20;
constexpr size_t CNT_OFF = CTRL_OFF;
constexpr size_t TMO_OFF = CTRL_OFF + 256;
constexpr size_t TICK_OFF = CTRL_OFF + 1024;
constexpr size_t G1_OFF = CTRL_OFF + 8192;
constexpr size_t G2_OFF = CTRL_OFF + 65536;
constexpr size_t DONE_OFF = CTRL_OFF + 196608;
constexpr size_t AV_OFF = CTRL_OFF + 262144;
constexpr int AV_LINK = 336 * AVS;
constexpr size_t WS_NEED = AV_OFF + 8 * AV_LINK;
static_assert((size_t)NSTREAM * STREAM_STRIDE <= CTRL_OFF, "layout");
constexpr int AUX_SYS = 17;
#define AUX_TK 16
constexpr int RSRC_W3 = 0x00020000;
constexpr unsigned long long DEADLINE_TICKS = 100000000ull;
constexpr unsigned long long TMO_LOOK_TICKS = 20000ull;

#define PROF_T(slot) do { } while (0)
#define PROF_V(slot, v) do { } while (0)
#define PROF_S(x)

struct Params {
  const float* x;
  const float* w;
  char* myws;
  unsigned rank, pad;
  char* ws[NR];
  char* out[NR];
};

constexpr unsigned long long src_table(int part) {
  unsigned long long t = 0;
  for (int i = 0; i < 16; ++i) {
    const int u = 16 * part + i;
    const int sub = u >= 35 ? u - 35 : u / 7, d = u >= 35 ? 0 : u % 7 + 1;
    const bool ux = d <= 1 ? sub < 5 : sub == 0;
    const int b = d <= 1 ? (ux ? sub : sub - 5) : (ux ? 4 + (d >> 1) : 4 * (d & 1) + sub - 1);
    t |= (unsigned long long)((ux ? 8 : 0) | b) << (4 * i);
  }
  return t;
}

__device__ __forceinline__ unsigned pk_bf16(float a, float b) {
  unsigned r;
  asm("v_cvt_pk_bf16_f32 %0, %1, %2" : "=v"(r) : "v"(a), "v"(b));
  return r;
}
__device__ __forceinline__ unsigned ld_sys32(const void* p) {
  return __hip_atomic_load((const unsigned*)p, __ATOMIC_RELAXED, __HIP_MEMORY_SCOPE_SYSTEM);
}
__device__ __forceinline__ unsigned long long ld_sys64(const void* p) {
  return __hip_atomic_load((const unsigned long long*)p, __ATOMIC_RELAXED, __HIP_MEMORY_SCOPE_SYSTEM);
}
__device__ __forceinline__ void st_sys32(void* p, unsigned v) {
  __hip_atomic_store((unsigned*)p, v, __ATOMIC_RELAXED, __HIP_MEMORY_SCOPE_SYSTEM);
}
__device__ __forceinline__ void st_sys64(void* p, unsigned long long v) {
  __hip_atomic_store((unsigned long long*)p, v, __ATOMIC_RELAXED, __HIP_MEMORY_SCOPE_SYSTEM);
}

__device__ __forceinline__ bool wait_expired(unsigned long long t0, char* myws, unsigned seq) {
  const unsigned long long el = __builtin_amdgcn_s_memrealtime() - t0;
  if (el > TMO_LOOK_TICKS) {
    unsigned* tmo = (unsigned*)(myws + TMO_OFF);
    const unsigned tv = __builtin_amdgcn_readfirstlane(
        __hip_atomic_load(tmo, __ATOMIC_RELAXED, __HIP_MEMORY_SCOPE_AGENT));
    if (tv == seq || el > DEADLINE_TICKS) {
      __hip_atomic_store(tmo, seq, __ATOMIC_RELAXED, __HIP_MEMORY_SCOPE_AGENT);
      return true;
    }
  }
  return false;
}

__device__ __forceinline__ unsigned wave_max_u(unsigned v) {
  v = max(v, (unsigned)__builtin_amdgcn_update_dpp(0, (int)v, 0x111, 0xf, 0xf, false));
  v = max(v, (unsigned)__builtin_amdgcn_update_dpp(0, (int)v, 0x112, 0xf, 0xf, false));
  v = max(v, (unsigned)__builtin_amdgcn_update_dpp(0, (int)v, 0x114, 0xf, 0xf, false));
  v = max(v, (unsigned)__builtin_amdgcn_update_dpp(0, (int)v, 0x118, 0xf, 0xf, false));
  v = max(v, (unsigned)__builtin_amdgcn_update_dpp(0, (int)v, 0x142, 0xa, 0xf, false));
  v = max(v, (unsigned)__builtin_amdgcn_update_dpp(0, (int)v, 0x143, 0xc, 0xf, false));
  return (unsigned)__builtin_amdgcn_readlane((int)v, 63);
}

}

__global__ void __launch_bounds__(256, 2) ag_gemm_fused(Params P) {
  extern __shared__ __attribute__((aligned(16))) char lds[];
  const int lane = threadIdx.x & 63;
  const int wv = __builtin_amdgcn_readfirstlane(threadIdx.x >> 6);
  const int blk = blockIdx.x;
  const unsigned rank = P.rank & 7u;
  char* const myws = P.myws;
  const float* px = P.x;
  const float* pw = P.w;
  asm volatile("" : "+s"(px), "+s"(pw));
  const int l15 = lane & 15, q = lane >> 4;
  PROF_T(0);

  const int xg = blk & 7, li = blk >> 3;
  const int ti = (li - NSB) * GPW + wv;
  if (__builtin_expect(li < NSB || wv >= GPW || ti >= 64, 1)) {
    int sw;
    if (li < NSB) {
      if (wv == 0) { sw = blk; if (sw >= NPSH) return; }
      else { sw = -2; if (wv != 1 || blk >= NSEND) return; }
    }
    else if (wv >= GPW) { sw = NPSH + (blk - 8 * NSB); if (wv > GPW || sw >= NPSH + NWLOC) return; }
    else { if (ti != 64) return; sw = -1; }
    if (__builtin_expect(sw >= 0, 1)) {
      const bool loc = sw >= NPSH;
      const int np = loc ? 16 * 64 : NPSH * NPUSH, magic = loc ? 4096 : 1873, um = loc ? 16 : 35, gs = loc ? NWLOC : NPSH;
      const int voffx = (q * KDIM + 4 * l15) * 4, voffw = ((4 * q) * NLOC + 4 * l15) * 4;
      char* const lw = lds + wv * 10240 + (l15 >> 3) * 5120 + (l15 & 3) * 320 + q * 16 + ((l15 >> 2) & 1) * 8;
      const char* const lr = lds + wv * 10240 + q * 320 + l15 * 16;
      bool isx;
      int ks, un;
      u32x4 ld[16], o[8];
#define DECODE_LOAD(g, valid) { \
        ks = ((g) * magic) >> 16; \
        un = (g) - ks * um; \
        if (loc && ks < KROT + 4) ks = ks < 4 ? ks + KROT : ks - 4; \
        const unsigned ts = (unsigned)((loc ? 0x76543210fedcba98ull : un < 16 ? src_table(0) : un < 32 ? src_table(1) : src_table(2)) >> (4 * (un & 15))); \
        isx = (ts & 8u) != 0; \
        const __amdgpu_buffer_rsrc_t rs = __builtin_amdgcn_make_buffer_rsrc((void*)(isx ? px : pw), 0, (valid) ? 512 * 4096 * 4 : 0, RSRC_W3); \
        const int vo_ = isx ? voffx : voffw; \
        const int so_ = (int)(ts & 7u) * (isx ? 64 * KDIM * 4 : 256) + ks * (isx ? 256 : 64 * NLOC * 4); \
        const int sa_ = isx ? 4 * KDIM * 4 : NLOC * 4, sc_ = isx ? 16 * KDIM * 4 : 16 * NLOC * 4; \
        _Pragma("unroll") for (int i = 0; i < 16; ++i) \
          ld[i] = __builtin_amdgcn_raw_buffer_load_b128(rs, vo_, so_ + (i & 3) * sa_ + (i >> 2) * sc_, 0); \
        asm volatile("" ::: "memory"); \
      }
#define CONVERT() { \
        if (isx) { \
          _Pragma("unroll") for (int i = 0; i < 16; ++i) { \
            const f32x4 a = __builtin_bit_cast(f32x4, ld[i]); \
            *(u32x2*)(lw + (i >> 2) * 1280 + (i & 3) * 64) = u32x2{pk_bf16(a.x, a.y), pk_bf16(a.z, a.w)}; \
          } \
          _Pragma("unroll") for (int i = 0; i < 8; ++i) o[i] = *(const u32x4*)(lr + i * 1280); \
        } else { \
          _Pragma("unroll") for (int kh = 0; kh < 2; ++kh) \
          _Pragma("unroll") for (int b = 0; b < 4; ++b) { \
            unsigned r[4]; \
            _Pragma("unroll") for (int jj = 0; jj < 4; ++jj) \
              r[jj] = pk_bf16(__builtin_bit_cast(f32x4, ld[kh * 8 + 2 * jj])[b], __builtin_bit_cast(f32x4, ld[kh * 8 + 2 * jj + 1])[b]); \
            o[kh * 4 + b] = u32x4{r[0], r[1], r[2], r[3]}; \
          } \
        } \
        dks = ks; dun = un; \
        dst = (char*)(((unsigned long long)(unsigned)__builtin_amdgcn_readlane((int)(tdst >> 32), un) << 32) | \
                      (unsigned)__builtin_amdgcn_readlane((int)tdst, un)); \
        asm volatile("" ::: "memory"); \
      }
      int g = loc ? sw - NPSH : sw, dks, dun, fks = 0, fun = -2;
      char* dst;
      if (CSLEEP && sw >= NPSH + 64) __builtin_amdgcn_s_sleep(CSLEEP);
      DECODE_LOAD(g, true);
      __builtin_amdgcn_sched_barrier(0);
      const unsigned cnt = __hip_atomic_load((const unsigned*)(myws + CNT_OFF), __ATOMIC_RELAXED, __HIP_MEMORY_SCOPE_AGENT);
      unsigned long long tdst;
      char* tflag;
      int tkey, tstep = 4, tmin = 0;
      {
        const int r1 = (int)(rank & 1);
        if (loc) {
          const int e = lane >> 2, i = lane & 3;
          const int d = i == 0 ? 0 : e < 5 ? 1 : e < 8 ? 2 * e + i - 9 : 2 * i + ((e >> 2) & 1);
          const int sub = e < 5 ? e : e < 8 ? 0 : (e & 3) + 1;
          const bool ok = e < 5 ? i < 2 : e < 8 ? i < 3 : true;
          tkey = ok ? e : -1;
          tstep = i == 0 ? 4 : 5 * AVS;
          if (i != 0) tmin = (T0 + 4 - sub) / 5;
          const unsigned off = i == 0 ? (unsigned)(e < 5 ? 5 * r1 + e : e < 8 ? 43 + e : 8 + e) * STREAM_STRIDE + STREAM_BYTES
                                      : (unsigned)AV_OFF + (unsigned)(((d - 1) & 7) * AV_LINK + sub * AVS);
          tflag = P.ws[(rank ^ (unsigned)d) & 7u] + off;
          tdst = (unsigned long long)myws + (unsigned)(l15 < 5 ? 5 * r1 + l15 : l15 < 8 ? 43 + l15 : 8 + l15) * STREAM_STRIDE;
        } else {
          const int u = lane < 35 ? lane : 34;
          const int sub = (u * 37) >> 8, d = u - 7 * sub + 1, j = (d >> 1) - 1;
          const int stream = d == 1 ? 5 * r1 + sub : (sub == 0 ? 10 + 2 * j + r1 : 23 + 4 * (2 * j + r1) + sub);
          tkey = lane;
          tdst = (unsigned long long)P.ws[(rank ^ (unsigned)d) & 7u] + (unsigned)stream * STREAM_STRIDE;
          tflag = (char*)tdst + STREAM_BYTES;
        }
      }
      CONVERT();
      const unsigned seq = __builtin_amdgcn_readfirstlane(cnt) + 1u;
#define STORES() { \
        const __amdgpu_buffer_rsrc_t rd = __builtin_amdgcn_make_buffer_rsrc(dst, 0, (int)STREAM_BYTES, RSRC_W3); \
        _Pragma("unroll") for (int i = 0; i < 8; ++i) \
          __builtin_amdgcn_raw_buffer_store_b128(o[i], rd, lane * 16 + (i & 3) * 1024, dks * 8192 + (i >> 2) * 4096, AUX_SYS); \
        asm volatile("" ::: "memory"); \
      }
      if (loc) {
        STORES();
        g += gs;
        DECODE_LOAD(g, true);
        asm volatile("s_waitcnt vmcnt(16)" ::: "memory");
        if (tkey == dun && dks >= tmin) st_sys32(tflag + dks * tstep, seq);
        PROF_T(2);
        CONVERT();
      }
      if (PSLEEP && !loc && sw >= 7 * PEARLY) __builtin_amdgcn_s_sleep(PSLEEP);
      bool more = g + gs < np;
      g = more ? g + gs : g;
      DECODE_LOAD(g, more);
      for (;;) {
        STORES();
        asm volatile("s_waitcnt vmcnt(24)" ::: "memory");
        if (tkey == fun && fks >= tmin) st_sys32(tflag + fks * tstep, seq);
        asm volatile("" ::: "memory");
        fun = dun; fks = dks;
        if (!more) break;
        CONVERT();
        more = g + gs < np;
        g = more ? g + gs : g;
        DECODE_LOAD(g, more);
      }
      asm volatile("s_waitcnt vmcnt(0)" ::: "memory");
      if (tkey == fun && fks >= tmin) st_sys32(tflag + fks * tstep, seq);
      PROF_T(1);
      return;
    }
    if (sw == -2) {
      const unsigned cnt = __hip_atomic_load((const unsigned*)(myws + CNT_OFF), __ATOMIC_RELAXED, __HIP_MEMORY_SCOPE_AGENT);
      const int w = (blk * 37) >> 8, dl = blk - 7 * w, d = dl + 1, j = (d >> 1) - 1;
      const unsigned sr = (rank ^ (unsigned)d) & 7u;
      const int s1 = (int)(sr & 1);
      char* const sws = P.ws[sr];
      const int src0 = d == 1 ? 5 * s1 : 48 + j, srcn = d == 1 ? 5 * s1 : 15 + 4 * (d & 1);
      const int dst0 = d == 1 ? 5 * s1 : 10 + 2 * j + s1, dstn = d == 1 ? 5 * s1 : 23 + 4 * (2 * j + s1);
      const int foff = lane * (1 << 20);
      const __amdgpu_buffer_rsrc_t rt = __builtin_amdgcn_make_buffer_rsrc(myws + TICK_OFF + dl * 256, 0, 4, RSRC_W3);
      const char* const av = myws + AV_OFF + dl * AV_LINK + min(lane, 15) * AVS;
      const unsigned seq = __builtin_amdgcn_readfirstlane(cnt) + 1u;
      const unsigned tbase = (unsigned)(T0 + 2 * NWL) - (seq - 1u) * (unsigned)(320 - T0 - NWL);
      u32x4 b0[8], b1[8];
      bool dead = false;
      unsigned avm = 0;
#define AVAIL(t, mask) { \
        const unsigned long long t0 = __builtin_amdgcn_s_memrealtime(); \
        while (!dead) { \
          const unsigned v = ld_sys32(av + (t) * AVS); \
          avm = (unsigned)__builtin_amdgcn_ballot_w64((int)(v - seq) >= 0); \
          if ((avm & (mask)) == (mask)) break; \
          dead = wait_expired(t0, myws, seq); \
          __builtin_amdgcn_s_sleep(1); \
        } \
        asm volatile("" ::: "memory"); \
      }
#define PULL(buf, t, valid) { \
        const int ks_ = ((t) * 13108) >> 16, sub_ = (t) - 5 * ks_; \
        const __amdgpu_buffer_rsrc_t rs = __builtin_amdgcn_make_buffer_rsrc( \
            sws + (unsigned)(sub_ == 0 ? src0 : srcn + sub_) * STREAM_STRIDE, 0, (valid) ? (int)STREAM_BYTES : 0, RSRC_W3); \
        _Pragma("unroll") for (int i = 0; i < 8; ++i) \
          buf[i] = __builtin_amdgcn_raw_buffer_load_b128(rs, lane * 16 + (i & 3) * 1024, ks_ * 8192 + (i >> 2) * 4096, AUX_SYS); \
        asm volatile("" ::: "memory"); \
      }
      int tc = T0 + w, tn = T0 + NWL + w, fsz = 0;
      char* fdst = myws;
      bool vn = true;
      AVAIL(tc, 1u);
      PROF_S(unsigned long long tB = __builtin_amdgcn_s_memrealtime(); )
      PULL(b0, tc, true);
      if (!((avm >> NWL) & 1u)) AVAIL(tn, 1u);
      unsigned tk = (unsigned)__builtin_amdgcn_raw_ptr_buffer_atomic_add_i32(1, rt, foff, 0, AUX_TK);
      unsigned avv = ld_sys32(av + tn * AVS);
      PULL(b1, tn, true);
#define PSTEP(buf) { \
        asm volatile("s_waitcnt vmcnt(8)" ::: "memory"); \
        __builtin_amdgcn_raw_buffer_store_b32(seq, __builtin_amdgcn_make_buffer_rsrc(fdst, 0, fsz, RSRC_W3), foff, 0, AUX_SYS); \
        PROF_S(__builtin_amdgcn_raw_buffer_store_b32((unsigned)__builtin_amdgcn_s_memrealtime(), __builtin_amdgcn_make_buffer_rsrc(myws + (41ull << 20) + (dl * 320 + tc) * 4, 0, 4, RSRC_W3), foff, 0, 0);) \
        const int ks_ = (tc * 13108) >> 16, sub_ = tc - 5 * ks_; \
        char* const dst = myws + (unsigned)(sub_ == 0 ? dst0 : dstn + sub_) * STREAM_STRIDE; \
        const __amdgpu_buffer_rsrc_t rd = __builtin_amdgcn_make_buffer_rsrc(dst, 0, (int)STREAM_BYTES, RSRC_W3); \
        _Pragma("unroll") for (int i = 0; i < 8; ++i) \
          __builtin_amdgcn_raw_buffer_store_b128(buf[i], rd, lane * 16 + (i & 3) * 1024, ks_ * 8192 + (i >> 2) * 4096, AUX_SYS); \
        asm volatile("" ::: "memory"); \
        fdst = dst + STREAM_BYTES + ks_ * 4; fsz = 4; \
        if (!vn) break; \
        const unsigned t2u = __builtin_amdgcn_readfirstlane(tk) + tbase; \
        vn = t2u < 320u; \
        const int t2 = vn ? (int)t2u : 319; \
        const unsigned long long m = __builtin_amdgcn_ballot_w64((int)(avv - seq) >= 0); \
        const unsigned rel = (unsigned)(t2 - tn); \
        if (vn && !(rel < 16u && ((m >> (rel & 15u)) & 1ull))) AVAIL(t2, 1u); \
        tk = (unsigned)__builtin_amdgcn_raw_ptr_buffer_atomic_add_i32(1, rt, vn ? foff : foff + (1 << 20), 0, AUX_TK); \
        avv = ld_sys32(av + t2 * AVS); \
        PULL(buf, t2, vn); \
        tc = tn; tn = t2; \
      }
      for (;;) { PSTEP(b0) PSTEP(b1) }
      asm volatile("s_waitcnt vmcnt(0)" ::: "memory");
      __builtin_amdgcn_raw_buffer_store_b32(seq, __builtin_amdgcn_make_buffer_rsrc(fdst, 0, 4, RSRC_W3), foff, 0, AUX_SYS);
      PROF_T(1);
      PROF_V(2, tB);
      return;
    }
    const unsigned cnt = __hip_atomic_load((const unsigned*)(myws + CNT_OFF), __ATOMIC_RELAXED, __HIP_MEMORY_SCOPE_AGENT);
    const unsigned seq = __builtin_amdgcn_readfirstlane(cnt) + 1u;
    {
      const char* g1 = myws + G1_OFF + (size_t)xg * FLAG_STRIDE + lane * 8;
      unsigned long long dsta = (unsigned long long)(P.ws[lane & 7] + G2_OFF +
          (size_t)((seq & 1u) * 8u + (unsigned)(lane >> 3)) * FLAG_STRIDE + (rank * 8u + (unsigned)xg) * 8u);
      asm volatile("" : "+v"(dsta));
      unsigned long long g;
      const unsigned long long t0 = __builtin_amdgcn_s_memrealtime();
      for (;;) {
        g = ld_sys64(g1);
        const unsigned long long m = __builtin_amdgcn_ballot_w64((unsigned)(g >> 32) == seq);
        if (m == ~0ull) break;
        if (wait_expired(t0, myws, seq)) break;
        if (__builtin_popcountll(m) < 32) __builtin_amdgcn_s_sleep(CLONG); else __builtin_amdgcn_s_sleep(1);
      }
      const unsigned m = wave_max_u((unsigned)g);
      __hip_atomic_store((__attribute__((address_space(1))) unsigned long long*)dsta, ((unsigned long long)seq << 32) | m,
                         __ATOMIC_RELAXED, __HIP_MEMORY_SCOPE_SYSTEM);
      PROF_T(1);
    }
    if (xg < 6) {
      const char* f = myws + DONE_OFF + (size_t)xg * FLAG_STRIDE + lane * 4;
      const unsigned long long skip = 15ull << (4 * (10 + (xg & 6) + (int)(rank & 1)));
      const unsigned long long t0 = __builtin_amdgcn_s_memrealtime();
      for (;;) {
        const unsigned v = ld_sys32(f);
        const unsigned long long m = __builtin_amdgcn_ballot_w64((int)(v - seq) >= 0) | skip;
        if (m == ~0ull) break;
        if (wait_expired(t0, myws, seq)) {
          if (lane == 0) __hip_atomic_store((signed char*)(P.out[rank] + (size_t)(rank & 6u) * 512 * NLOC), (signed char)-128,
                                            __ATOMIC_RELAXED, __HIP_MEMORY_SCOPE_SYSTEM);
          break;
        }
        if (__builtin_popcountll(m) < 24) __builtin_amdgcn_s_sleep(DLONG); else __builtin_amdgcn_s_sleep(1);
      }
    }
    PROF_T(2);
    return;
  }

  const unsigned cnt = __hip_atomic_load((const unsigned*)(myws + CNT_OFF), __ATOMIC_RELAXED, __HIP_MEMORY_SCOPE_AGENT);
  int rb = 8 * (xg & 1) + (ti >> 3);
  int cb = 4 * (ti & 7) + (xg >> 1);
  if (rb >= 10 && cb >= 8 && rb - 10 == (cb - 8) >> 2) { cb = (cb - 8) & 7; rb = 48 + ((rb - 10) >> 1); }
  const __amdgpu_buffer_rsrc_t rA =
      __builtin_amdgcn_make_buffer_rsrc(myws + (size_t)rb * STREAM_STRIDE, 0, (int)STREAM_BYTES, RSRC_W3);
  const __amdgpu_buffer_rsrc_t rB =
      __builtin_amdgcn_make_buffer_rsrc(myws + (size_t)(16 + cb) * STREAM_STRIDE, 0, (int)STREAM_BYTES, RSRC_W3);
  const char* const fl = myws + (size_t)(lane < 32 ? rb : 16 + cb) * STREAM_STRIDE + STREAM_BYTES + (lane & 31) * 4;
  const unsigned seq = __builtin_amdgcn_readfirstlane(cnt) + 1u;
  const int vo = lane * 16;

  int ready = 0;
  bool dead = false;
  f32x4 acc[4][4];
#pragma unroll
  for (int i = 0; i < 4; ++i)
#pragma unroll
    for (int j = 0; j < 4; ++j) acc[i][j] = f32x4{0.f, 0.f, 0.f, 0.f};
  u32x4 fx[5][4], fw[5][4];

#define POLL(need) \
  if (ready <= (need)) { \
    const unsigned long long t0 = __builtin_amdgcn_s_memrealtime(); \
    for (;;) { \
      const int base = ready & 32; \
      const unsigned v = ld_sys32(fl + base * 4); \
      const unsigned long long m = __builtin_amdgcn_ballot_w64((int)(v - seq) >= 0); \
      const unsigned both = (unsigned)m & (unsigned)(m >> 32); \
      const int c = (both == 0xffffffffu) ? 32 : __builtin_ctz(~both); \
      ready = base + c; \
      if (ready > (need)) break; \
      if (c == 32) continue; \
      if (wait_expired(t0, myws, seq)) { dead = true; ready = 64; break; } \
      __builtin_amdgcn_s_sleep(POLL_SLEEP); \
    } \
  } \
  asm volatile("" ::: "memory");

#define LOADH(slot, h) \
  _Pragma("unroll") for (int i = 0; i < 4; ++i) { \
    fw[slot][i] = __builtin_amdgcn_raw_buffer_load_b128(rB, vo + i * 1024, (h) * 4096, AUX_LD); \
    fx[slot][i] = __builtin_amdgcn_raw_buffer_load_b128(rA, vo + i * 1024, (h) * 4096, AUX_LD); \
  } \
  __builtin_amdgcn_sched_barrier(0);

#define MMA(slot) \
  _Pragma("unroll") for (int b = 0; b < 4; ++b) \
  _Pragma("unroll") for (int mb = 0; mb < 4; ++mb) \
    asm volatile("v_mfma_f32_16x16x32_bf16 %0, %1, %2, %0" : "+v"(acc[b][mb]) : "v"(fw[slot][b]), "v"(fx[slot][mb])); \
  __builtin_amdgcn_sched_barrier(0);

  POLL(0); LOADH(0, 0); LOADH(1, 1);
  PROF_T(5);
  POLL(1); LOADH(2, 2); LOADH(3, 3);
  for (int h = 0; h < 125; h += 5) {
#define STEP(u) { const int hn = (h + u + 4 < 127) ? h + u + 4 : 127; POLL(hn >> 1); LOADH((u + 4) % 5, hn); MMA(u); }
    STEP(0) STEP(1) STEP(2) STEP(3) STEP(4)
  }
  MMA(0) MMA(1) MMA(2)
  asm volatile("s_nop 7\n\ts_nop 7" : "+v"(acc[0][0]), "+v"(acc[0][1]), "+v"(acc[0][2]), "+v"(acc[0][3]),
               "+v"(acc[1][0]), "+v"(acc[1][1]), "+v"(acc[1][2]), "+v"(acc[1][3]), "+v"(acc[2][0]), "+v"(acc[2][1]),
               "+v"(acc[2][2]), "+v"(acc[2][3]), "+v"(acc[3][0]), "+v"(acc[3][1]), "+v"(acc[3][2]), "+v"(acc[3][3]) :: "memory");

  float mx = 0.f;
#pragma unroll
  for (int b = 0; b < 4; ++b)
#pragma unroll
    for (int mb = 0; mb < 4; ++mb)
#pragma unroll
      for (int r = 0; r < 4; ++r) mx = fmaxf(mx, acc[b][mb][r]);
  const unsigned mxu = wave_max_u(__builtin_bit_cast(unsigned, mx));
  if (lane == 0)
    st_sys64(myws + G1_OFF + (size_t)xg * FLAG_STRIDE + ti * 8, ((unsigned long long)seq << 32) | mxu);

  PROF_T(1);
  const unsigned pr = rank >> 1;
  unsigned owner, col0, row0, dslot = 0;
  if (cb < 8) { owner = rank; col0 = 64 * cb; }
  else {
    const unsigned u = (cb - 8) >> 2, c = (cb - 8) & 3, j = u >> 1;
    owner = (((pr ^ (j + 1)) << 1) | (u & 1)) & 7u;
    col0 = 256 * ((rank ^ owner) & 1) + 64 * c;
    dslot = (2 * j + (rank & 1)) * FLAG_STRIDE + (rb * 4 + c) * 4;
  }
  if (rb < 10) row0 = 512 * (2 * pr + (rb >= 5)) + 64 * (rb >= 5 ? rb - 5 : rb);
  else if (rb >= 48) row0 = 512 * rank + 320 + 64 * (rb - 48);
  else {
    const unsigned u = rb - 10, j = u >> 1;
    row0 = 512 * ((((pr ^ (j + 1)) << 1) | (u & 1)) & 7u) + 320 + 64 * j;
  }
  const __amdgpu_buffer_rsrc_t rO = __builtin_amdgcn_make_buffer_rsrc(P.out[owner], 0, KDIM * NLOC, RSRC_W3);

  float gm;
  {
    const char* g2 = myws + G2_OFF + (size_t)((seq & 1u) * 8u + (unsigned)xg) * FLAG_STRIDE + lane * 8;
    const unsigned long long t0 = __builtin_amdgcn_s_memrealtime();
    unsigned long long g;
    const unsigned tv = __hip_atomic_load((const unsigned*)(myws + TMO_OFF), __ATOMIC_RELAXED, __HIP_MEMORY_SCOPE_AGENT);
    for (;;) {
      g = ld_sys64(g2);
      const unsigned long long m = __builtin_amdgcn_ballot_w64((unsigned)(g >> 32) == seq);
      if (m == ~0ull) break;
      if (wait_expired(t0, myws, seq)) { dead = true; break; }
      if (__builtin_popcountll(m) < G2CNT) __builtin_amdgcn_s_sleep(G2LONG); else __builtin_amdgcn_s_sleep(G2SLEEP);
    }
    if (__builtin_amdgcn_readfirstlane(tv) == seq) dead = true;
    gm = __builtin_bit_cast(float, wave_max_u((unsigned)g));
  }
  PROF_T(2);
  if (xg == 0 && ti == 0 && lane == 0)
    __hip_atomic_store((unsigned*)(myws + CNT_OFF), seq, __ATOMIC_RELAXED, __HIP_MEMORY_SCOPE_AGENT);
  const float inv = gm > 0.f ? 127.0f / gm : 0.f;
#pragma unroll
  for (int mb = 0; mb < 4; ++mb) {
    u32x4 o = u32x4{0u, 0u, 0u, 0u};
#pragma unroll
    for (int b = 0; b < 4; ++b) {
      const f32x4 t = acc[b][mb] * inv;
#pragma unroll
      for (int r = 0; r < 4; ++r) o[r] = __builtin_amdgcn_cvt_pk_u8_f32(__builtin_rintf(t[r]), b, o[r]);
    }
    if (dead) o = u32x4{0x80808080u, 0x80808080u, 0x80808080u, 0x80808080u};
    __builtin_amdgcn_raw_buffer_store_b128(o, rO, (int)((row0 + l15) * NLOC + col0 + 16 * q), mb * 16 * NLOC, AUX_SYS);
    __builtin_amdgcn_sched_barrier(0);
  }
  if (cb >= 8) {
    asm volatile("s_waitcnt vmcnt(0)" ::: "memory");
    if (lane == 0) st_sys32(P.ws[owner] + DONE_OFF + dslot, seq);
  }
  PROF_T(3);
}

extern "C" void kernel_launch(int rank, int nranks, ncclComm_t comm, hipStream_t stream,
                              void* const* d_in, const int* in_sizes, int n_in,
                              void* d_out, int out_size,
                              const int* in_dtypes, int out_dtype,
                              void* const* d_out_all,
                              void* const* d_ws_all, size_t ws_bytes) {
  (void)comm; (void)in_dtypes; (void)out_dtype; (void)d_out;
  if (nranks != NR || (unsigned)rank >= (unsigned)NR || n_in < 2 || ws_bytes < WS_NEED ||
      in_sizes[0] != 512 * KDIM || in_sizes[1] != KDIM * NLOC || out_size != KDIM * NLOC) {
    fprintf(stderr, "unexpected problem shape\n");
    return;
  }
  Params p;
  p.x = (const float*)d_in[0];
  p.w = (const float*)d_in[1];
  for (int i = 0; i < NR; ++i) { p.ws[i] = (char*)d_ws_all[i]; p.out[i] = (char*)d_out_all[i]; }
  p.myws = p.ws[rank];
  p.rank = (unsigned)rank;
  p.pad = 0;
  ag_gemm_fused<<<256, 256, LDS_PAD, stream>>>(p);
  HIP_CHECK(hipGetLastError());
}
```
